```python
import numpy as np

kernel_name = "gpu_gemm_a2a_m4096_k4096_n4096_w8a8_dequant"


ABSMAX_RTOL = 0.001

COMPUTE_DTYPE = "int8"
EPILOGUE = "dequant"
IN_QUANT = "int8"
OUT_QUANT = None
IN_DTYPES = {"x": "int8", "w_mat": "int8",
             "scale_x": "float32", "scale_w": "float32"}
WORLD_SIZE = 8

def setup_inputs():

    x = np.random.randint(-127, 128, size=(4096, 4096)).astype(np.int8)
    w_mat = np.random.randint(-127, 128, size=(4096, 4096)).astype(np.int8)
    scale_x = (0.02 * (1.0 + np.random.rand(1))).astype(np.float32)
    scale_w = (0.02 * (1.0 + np.random.rand(1))).astype(np.float32)
    return {"x": x, "w_mat": w_mat, "scale_x": scale_x, "scale_w": scale_w}

def reference(x, w_mat, scale_x, scale_w):
    acc = x.astype(np.float64) @ w_mat.astype(np.float64)
    y = (acc.astype(np.float32) * (scale_x * scale_w)).astype(np.float32)
    return y
```

```cpp
#include <hip/hip_runtime.h>
#include <rccl/rccl.h>
#include <cstdio>
#include <cstdint>

typedef int v2i __attribute__((ext_vector_type(2)));
typedef int v4i __attribute__((ext_vector_type(4)));
typedef unsigned v4u __attribute__((ext_vector_type(4)));
typedef float v4f __attribute__((ext_vector_type(4)));
#define LDS_AS __attribute__((address_space(3)))
#define CONST_AS __attribute__((address_space(4)))

constexpr int NR = 8;
constexpr int ML = 512;
constexpr int KK = 4096;
constexpr int NN = 4096;
constexpr int NC = 512;
constexpr int NWG = 256;
constexpr int NT = 512;
constexpr int WAVE_LDS = 16384;
constexpr int LDS_BYTES = 8 * WAVE_LDS;
constexpr unsigned long long DEADLINE_TICKS = 20000000ull;

constexpr size_t WS_CNT = 0;
constexpr size_t WS_TMO = WS_CNT + NWG * 128;
constexpr size_t WS_FLAG = WS_TMO + 128;
constexpr size_t WS_DBG = WS_FLAG + NR * 32 * 128;
constexpr size_t WS_END = WS_DBG + NWG * 256;

struct Params {
  const int8_t* x;
  const int8_t* w;
  const float* sx;
  const float* sw;
  float* out[NR];
  char* ws[NR];
  int rank;
  int pad;
};

__device__ __forceinline__ v4u make_rsrc(const void* p, unsigned bytes) {
  unsigned long long a = (unsigned long long)p;
  v4u r;
  r[0] = __builtin_amdgcn_readfirstlane((unsigned)a);
  r[1] = __builtin_amdgcn_readfirstlane((unsigned)(a >> 32) & 0xffffu);
  r[2] = __builtin_amdgcn_readfirstlane(bytes);
  r[3] = 0x00020000u;
  return r;
}

#define STAMP(i) do {} while (0)

#define WAITV(n) asm volatile("s_waitcnt vmcnt(" #n ")" ::: "memory")

__device__ __forceinline__ void dma16(v4u rsrc, unsigned voff, unsigned soff, unsigned ldsaddr) {
  asm volatile("s_mov_b32 m0, %3\n\ts_nop 0\n\tbuffer_load_dwordx4 %0, %1, %2 offen lds"
               :: "v"(voff), "s"(rsrc), "s"(soff), "s"(ldsaddr) : "memory");
}
__device__ __forceinline__ void bload16(v4i& d, v4u rsrc, unsigned voff, unsigned soff) {
  asm volatile("buffer_load_dwordx4 %0, %1, %2, %3 offen" : "=v"(d) : "v"(voff), "s"(rsrc), "s"(soff) : "memory");
}

__global__ void __launch_bounds__(NT, 2) gemm_a2a(Params p) {
  __shared__ __attribute__((aligned(16))) char lds[LDS_BYTES];
  const int tid = threadIdx.x;
  const int lane = tid & 63;
  const int wave = __builtin_amdgcn_readfirstlane(tid >> 6);
  const int b = blockIdx.x;
  const int rank = p.rank;
  STAMP(0);
  const int xcd = b & 7, idx = b >> 3;
  const int rg = idx >> 3;
  const int dsel = ((xcd >> 2) << 2) | ((idx >> 1) & 3);
  const int cgl = ((xcd & 3) << 1) | (idx & 1);
  const int dst = dsel ^ rank;
  const int slot = rg * 8 + cgl;
  const int col0 = dst * NC + cgl * 64;
  const int row0 = rg * 128;

  const unsigned call = *(const CONST_AS unsigned*)(p.ws[rank] + WS_CNT + b * 128) + 1u;
  const float scale = *(const CONST_AS float*)p.sx * *(const CONST_AS float*)p.sw;

  const v4u rw = make_rsrc(p.w, (unsigned)KK * NN);
  const v4u rx = make_rsrc(p.x, (unsigned)ML * KK);
  const unsigned lbase = (unsigned)(uintptr_t)(LDS_AS char*)lds + wave * WAVE_LDS;
  const int g = lane >> 4;

  unsigned kof[8];
#pragma unroll
  for (int s = 0; s < 8; ++s) kof[s] = wave * 512 + ((s + 2 * rg) & 7) * 64;

  const unsigned wv0 = (lane >> 2) * NN + col0 + 16 * ((lane & 3) ^ ((lane >> 4) & 1));
  const unsigned wv1 = (lane >> 2) * NN + col0 + 16 * ((lane & 3) ^ ((lane >> 4) & 1) ^ 2);
  auto stageW = [&](int s) {
#pragma unroll
    for (int i = 0; i < 4; ++i)
      dma16(rw, (i & 1) ? wv1 : wv0, (kof[s] + 16 * i) * NN, lbase + (s & 3) * 4096 + i * 1024);
  };
  unsigned tra[4];
  {
    const int q = (lane & 15) >> 1, pp = lane & 1;
    const int f = ((q >> 2) & 1) | ((g & 1) << 1);
#pragma unroll
    for (int t = 0; t < 4; ++t) tra[t] = lbase + (16 * g + q) * 64 + ((t ^ f) * 16) + 8 * pp;
  }
  v4i Wf[8][4];
  auto readW = [&](int s) {
#pragma unroll
    for (int t = 0; t < 4; ++t) {
      v2i lo = __builtin_amdgcn_ds_read_tr8_b64_v2i32((LDS_AS v2i*)(tra[t] + (s & 3) * 4096));
      v2i hi = __builtin_amdgcn_ds_read_tr8_b64_v2i32((LDS_AS v2i*)(tra[t] + (s & 3) * 4096 + 512));
      Wf[s][t] = v4i{lo[0], lo[1], hi[0], hi[1]};
    }
  };
  const unsigned xv = (row0 + (lane & 15)) * KK + 16 * g;
  v4i Xf[2][8];
  auto loadX = [&](int j) {
#pragma unroll
    for (int s = 0; s < 8; ++s) bload16(Xf[j & 1][s], rx, xv, kof[s] + j * 16 * KK);
  };
#define PINX(bf) asm volatile("" : "+v"(Xf[bf][0]), "+v"(Xf[bf][1]), "+v"(Xf[bf][2]), "+v"(Xf[bf][3]), \
                                   "+v"(Xf[bf][4]), "+v"(Xf[bf][5]), "+v"(Xf[bf][6]), "+v"(Xf[bf][7]))
#define LGKM0 asm volatile("s_waitcnt lgkmcnt(0)" ::: "memory")

  const unsigned lds0 = (unsigned)(uintptr_t)(LDS_AS char*)lds;
  unsigned pwa[4];
#pragma unroll
  for (int t = 0; t < 4; ++t) { const int c = 4 * t + g; pwa[t] = lbase + (c * 16 + ((lane & 15) ^ c)) * 16; }
  const int rrow = 4 * (wave & 3) + g, rc = lane & 15;
  const unsigned pra = lds0 + (rc * 16 + (rrow ^ rc)) * 16;
  char* const obase = (char*)p.out[dst] + ((size_t)(rank * ML + row0) * NC + cgl * 64) * 4;
  const unsigned ov = (rrow * NC + 4 * rc) * 4;

  v4i acc[4];
  auto mma = [&](int j, int s) {
#pragma unroll
    for (int t = 0; t < 4; ++t) acc[t] = __builtin_amdgcn_mfma_i32_16x16x64_i8(Wf[s][t], Xf[j & 1][s], acc[t], 0, 0, 0);
  };
  auto finish = [&](int j) {
#pragma unroll
    for (int t = 0; t < 4; ++t) *(LDS_AS v4i*)(pwa[t] + (j & 1) * 4096) = acc[t];
    LGKM0;
    __builtin_amdgcn_s_barrier();
    asm volatile("" ::: "memory");
    if ((wave >> 2) == (j & 1)) {
      v4i sum = *(LDS_AS v4i*)(pra + (j & 1) * 4096);
#pragma unroll
      for (int w2 = 1; w2 < 8; ++w2) sum += *(LDS_AS v4i*)(pra + (j & 1) * 4096 + w2 * WAVE_LDS);
      v4f y;
#pragma unroll
      for (int r = 0; r < 4; ++r) y[r] = (float)sum[r] * scale;
      asm volatile("global_store_dwordx4 %0, %1, %2 sc0 sc1\n\ts_nop 1"
                   :: "v"(ov), "v"(y), "s"(obase + (size_t)j * 16 * NC * 4) : "memory");
    }
  };

  stageW(0); stageW(1); stageW(2); stageW(3);
  loadX(0);
  WAITV(20); readW(0); LGKM0; stageW(4);
  WAITV(20); readW(1); LGKM0; stageW(5);
  WAITV(20); readW(2); LGKM0; stageW(6);
  WAITV(20); readW(3); LGKM0; stageW(7);
  loadX(1);
  STAMP(1);
#pragma unroll
  for (int t = 0; t < 4; ++t) acc[t] = v4i{0, 0, 0, 0};
  WAITV(20); PINX(0); readW(4);
  mma(0, 0); mma(0, 1); mma(0, 2); mma(0, 3);
  WAITV(16); readW(5); mma(0, 4);
  WAITV(12); readW(6); mma(0, 5);
  WAITV(8); readW(7); mma(0, 6); mma(0, 7);
  LGKM0;
  STAMP(2);
  loadX(2);
  finish(0);
  STAMP(3);
#pragma unroll
  for (int j = 1; j < 8; ++j) {
    if (j == 1) WAITV(8);
    else if (j == 7) WAITV(1);
    else WAITV(9);
    PINX(j & 1);
#pragma unroll
    for (int t = 0; t < 4; ++t) acc[t] = v4i{0, 0, 0, 0};
#pragma unroll
    for (int s = 0; s < 8; ++s) mma(j, s);
    if (j + 2 < 8) {
      asm volatile("" :: "v"(acc[0]), "v"(acc[1]), "v"(acc[2]), "v"(acc[3]) : "memory");
      loadX(j + 2);
    }
    finish(j);
    STAMP(3 + j);
  }
  WAITV(0);
  __builtin_amdgcn_s_barrier();
  STAMP(11);
  if (tid == 0) {
    char* const myws = p.ws[rank];
    if (dst != rank) {
      __hip_atomic_store((unsigned*)(p.ws[dst] + WS_FLAG + (rank * 32 + slot) * 128), call, __ATOMIC_RELAXED, __HIP_MEMORY_SCOPE_SYSTEM);
      unsigned* flag = (unsigned*)(myws + WS_FLAG + (dst * 32 + slot) * 128);
      unsigned* tmo = (unsigned*)(myws + WS_TMO);
      const unsigned long long t0 = __builtin_amdgcn_s_memrealtime();
      while (__hip_atomic_load(flag, __ATOMIC_RELAXED, __HIP_MEMORY_SCOPE_SYSTEM) < call) {
        if (__hip_atomic_load(tmo, __ATOMIC_RELAXED, __HIP_MEMORY_SCOPE_AGENT) == call ||
            __builtin_amdgcn_s_memrealtime() - t0 > DEADLINE_TICKS) {
          __hip_atomic_store(tmo, call, __ATOMIC_RELAXED, __HIP_MEMORY_SCOPE_AGENT);
          break;
        }
        __builtin_amdgcn_s_sleep(1);
      }
      if (__hip_atomic_load(tmo, __ATOMIC_RELAXED, __HIP_MEMORY_SCOPE_AGENT) == call)
        p.out[rank][(size_t)(rank * ML) * NC] = __builtin_nanf("");
    }
    __hip_atomic_store((unsigned*)(myws + WS_CNT + b * 128), call, __ATOMIC_RELAXED, __HIP_MEMORY_SCOPE_AGENT);
  }
}

extern "C" void kernel_launch(int rank, int nranks, ncclComm_t comm, hipStream_t stream,
                              void* const* d_in, const int* in_sizes, int n_in, void* d_out, int out_size,
                              const int* in_dtypes, int out_dtype, void* const* d_out_all,
                              void* const* d_ws_all, size_t ws_bytes) {
  if (nranks != NR || n_in != 4 || (unsigned)rank >= (unsigned)NR || in_sizes[0] != ML * KK || in_sizes[1] != KK * NN ||
      out_size != NR * ML * NC || ws_bytes < WS_END) {
    fprintf(stderr, "kernel_launch: unexpected problem shape\n");
    return;
  }
  Params p;
  p.x = (const int8_t*)d_in[0];
  p.w = (const int8_t*)d_in[1];
  p.sx = (const float*)d_in[2];
  p.sw = (const float*)d_in[3];
  for (int i = 0; i < NR; ++i) { p.out[i] = (float*)d_out_all[i]; p.ws[i] = (char*)d_ws_all[i]; }
  p.rank = rank;
  p.pad = 0;
  gemm_a2a<<<NWG, NT, 0, stream>>>(p);
}
```

```cpp
#include <hip/hip_runtime.h>
#include <rccl/rccl.h>
#include <cstdio>
#include <cstdint>
#include <cstddef>

typedef int v2i __attribute__((ext_vector_type(2)));
typedef int v4i __attribute__((ext_vector_type(4)));
typedef unsigned v4u __attribute__((ext_vector_type(4)));
typedef float v4f __attribute__((ext_vector_type(4)));
#define LDS_AS __attribute__((address_space(3)))
#define CONST_AS __attribute__((address_space(4)))
typedef __attribute__((address_space(1))) unsigned gu32;

constexpr int NR = 8;
constexpr int ML = 512;
constexpr int KK = 4096;
constexpr int NN = 4096;
constexpr int NC = 512;
constexpr int NWG = 256;
constexpr int NT = 512;
constexpr int XR = 80;
constexpr int TM = 48;
constexpr int NRG = (ML - XR) / TM;
constexpr int NTILE = 4 * NRG;
static_assert(NRG * TM + XR == ML && NRG == 9 && XR == 80, "rows");
constexpr int NXW = 4, XPW = 4, NWK = 32 - NXW;
constexpr int XD = 4;
constexpr int NXP = XR * KK / 1024 / (NXW * XPW);
static_assert(NXP * NXW * XPW * 1024 == XR * KK && NXP % 2 == 0 && NXP / 2 + XD < NXP && XPW % 2 == 0, "x push");
constexpr int NSPL = 4;

constexpr int NST = KK / 128;
constexpr int STAGE = 16384 + 96 * 128;
constexpr int LAG = 2;
constexpr int LDS_T = (LAG + 2) * STAGE;
constexpr int LDS_C = LDS_T + 512;
constexpr int LDS_BYTES = LDS_T + 96 * 512;
static_assert(LDS_BYTES <= 163840, "lds");
constexpr unsigned long long DEADLINE_TICKS = 20000000ull;

constexpr size_t WS_CNT = 0;
constexpr size_t WS_TMO = WS_CNT + NWG * 128;
constexpr size_t WS_BF = WS_TMO + 128;
constexpr size_t WS_DBG = WS_BF + NR * NSPL * 4 * 128;
constexpr size_t WS_PB = 256u << 10;
constexpr size_t SLAB = 2u << 20;
constexpr size_t SL_TF = 0;
constexpr size_t SL_XF = SL_TF + NTILE * 4 * 128;
constexpr size_t SL_XS = 32u << 10;
constexpr size_t PK_TILE = 16384;
constexpr size_t SL_PK = SL_XS + (size_t)XR * KK;
constexpr size_t SL_PAR = (size_t)XR * KK + NTILE * PK_TILE;
constexpr size_t WS_END = NR * SLAB;
static_assert(WS_DBG + NWG * 256 + NR * 256 * 16 <= WS_PB && WS_PB + (size_t)NR * NSPL * 4 * 6144 <= SLAB &&
              SL_XF + 32 * 128 <= SL_XS && SL_XS + 2 * SL_PAR <= SLAB, "layout");

constexpr int PFM0 = 12, PFM1 = NWK - 6;
struct Peer {
  char* ws;
  char* out;
  const char* w;
  int dst, nsa;
};
struct Sched {
  unsigned short start, traw, t2, pad;
  unsigned short row0, col0;
  unsigned char mt, kind, unit, umem;
};
struct Params {
  const int8_t* x;
  const int8_t* w;
  const float* sx;
  const float* sw;
  char* myws;
  float* myout;
  int rank;
  int nmem, lgo, pad;
  Peer peer[NR];
  Sched sched[32];
  unsigned short pf[NWK - PFM0][8][8];
  unsigned short pf2[NWK - PFM1][8][8];
  unsigned uinfo[NTILE];
};

__device__ __forceinline__ v4u make_rsrc(const void* p, unsigned bytes) {
  unsigned long long a = (unsigned long long)p;
  v4u r;
  r[0] = __builtin_amdgcn_readfirstlane((unsigned)a);
  r[1] = __builtin_amdgcn_readfirstlane((unsigned)(a >> 32) & 0xffffu);
  r[2] = __builtin_amdgcn_readfirstlane(bytes);
  r[3] = 0x00020000u;
  asm volatile("s_nop 4" : "+s"(r));
  return r;
}

__device__ __forceinline__ v4u make_rsrc_s(const void* p, unsigned bytes) {
  const unsigned long long a = (unsigned long long)p;
  return v4u{(unsigned)a, (unsigned)(a >> 32) & 0xffffu, bytes, 0x00020000u};
}

#define DBG_PARAM
#define DBG_ARG
#define STAMPV(i) do {} while (0)

template <int N> __device__ __forceinline__ void waitv() { asm volatile("s_waitcnt vmcnt(%0)" :: "n"(N) : "memory"); }
#define LGKM0 asm volatile("s_waitcnt lgkmcnt(0)" ::: "memory")
#define BARRIER() do { asm volatile("" ::: "memory"); __builtin_amdgcn_s_barrier(); asm volatile("" ::: "memory"); } while (0)

__device__ __forceinline__ void dma16(v4u rsrc, unsigned voff, unsigned soff, unsigned ldsaddr) {
  asm volatile("s_mov_b32 m0, %3\n\ts_nop 0\n\tbuffer_load_dwordx4 %0, %1, %2 offen lds"
               :: "v"(voff), "s"(rsrc), "s"(soff), "s"(ldsaddr) : "memory");
}
__device__ __forceinline__ void dma16_sys(v4u rsrc, unsigned voff, unsigned soff, unsigned ldsaddr) {
  asm volatile("s_mov_b32 m0, %3\n\ts_nop 0\n\tbuffer_load_dwordx4 %0, %1, %2 offen sc0 sc1 lds"
               :: "v"(voff), "s"(rsrc), "s"(soff), "s"(ldsaddr) : "memory");
}
template <typename T> __device__ __forceinline__ void st16(const void* base, unsigned voff, T d) {
  asm volatile("global_store_dwordx4 %0, %1, off sc0 sc1\n\ts_nop 1" :: "v"((const char*)base + voff), "v"(d) : "memory");
}
__device__ __forceinline__ void st_flag(const void* addr, unsigned v) {
  asm volatile("global_store_dword %0, %1, off sc0 sc1" :: "v"(addr), "v"(v) : "memory");
}
__device__ __forceinline__ bool poll_ge(const char* addr, unsigned want, gu32* tmo, unsigned call) {
  gu32* const flag = (gu32*)addr;
  const unsigned long long t0 = __builtin_amdgcn_s_memrealtime();
  for (unsigned n = 1;; ++n) {
    const unsigned v = __hip_atomic_load(flag, __ATOMIC_RELAXED, __HIP_MEMORY_SCOPE_SYSTEM);
    if (__builtin_amdgcn_ballot_w64(v < want) == 0) return true;
    if ((n & 63) == 0 &&
        (__builtin_amdgcn_s_memrealtime() - t0 > DEADLINE_TICKS ||
         __builtin_amdgcn_readfirstlane(__hip_atomic_load(tmo, __ATOMIC_RELAXED, __HIP_MEMORY_SCOPE_AGENT)) == call)) {
      __hip_atomic_store(tmo, call, __ATOMIC_RELAXED, __HIP_MEMORY_SCOPE_AGENT);
      return false;
    }
    __builtin_amdgcn_s_sleep(1);
  }
}
__device__ __forceinline__ v4f dequant(v4i a, float s) {
  v4f y;
#pragma unroll
  for (int r = 0; r < 4; ++r) y[r] = (float)a[r] * s;
  return y;
}
__device__ __forceinline__ unsigned perm(unsigned hi, unsigned lo, unsigned sel) { return __builtin_amdgcn_perm(hi, lo, sel); }
__device__ __forceinline__ unsigned bfi(unsigned m, unsigned a, unsigned b) { return (a & m) | (b & ~m); }

__device__ __forceinline__ v4u pack12_lo(const v4i q[3]) {
  v4u d0;
  d0[0] = perm(q[0][1], q[0][0], 0x05040100u); d0[1] = perm(q[0][3], q[0][2], 0x05040100u);
  d0[2] = perm(q[1][1], q[1][0], 0x05040100u); d0[3] = perm(q[1][3], q[1][2], 0x05040100u);
  return d0;
}
__device__ __forceinline__ bool pack12_hi(const v4i q[3], v4u& d1) {
  unsigned H[3];
  int mx = q[0][0], mn = q[0][0];
#pragma unroll
  for (int i = 0; i < 3; ++i) {
    const unsigned h01 = perm(q[i][1], q[i][0], 0x0c0c0602u), h23 = perm(q[i][3], q[i][2], 0x06020c0cu);
    H[i] = h01 | h23;
#pragma unroll
    for (int j = 0; j < 4; ++j) { mx = max(mx, q[i][j]); mn = min(mn, q[i][j]); }
  }
  d1[0] = perm(q[2][1], q[2][0], 0x05040100u); d1[1] = perm(q[2][3], q[2][2], 0x05040100u);
  const unsigned E = (((H[0] ^ (H[0] << 1)) & 0x20202020u) >> 4) | (((H[1] ^ (H[1] << 1)) & 0x20202020u) >> 3) |
                     (((H[2] ^ (H[2] << 1)) & 0x20202020u) >> 2);
  const bool esc = (mx >= (1 << 21)) | (mn < -(1 << 21)) | ((E & (E - 1)) != 0);
  const unsigned pos = (unsigned)__builtin_ctz(E | 0x80000000u);
  unsigned code = ((pos >> 3) << 2) | (pos & 3);
  code = E ? code : 0u;
  code = esc ? 4u : code;
  d1[2] = bfi(0x1f1f1f1fu, H[0], H[2] << 5);
  d1[3] = bfi(0x7f7f7f7fu, bfi(0x1f1f1f1fu, H[1], H[2] << 2), code * 0x10204080u);
  return esc;
}
__device__ __forceinline__ bool unpack12(v4u d0, v4u d1, v4i q[3]) {
  const unsigned cb = d1[3] & 0x80808080u;
  const unsigned code = ((cb >> 7) | (cb >> 14) | (cb >> 21) | (cb >> 28)) & 15u;
  unsigned H[3];
  H[0] = d1[2]; H[1] = d1[3];
  H[2] = ((d1[2] >> 5) & 0x07070707u) | ((d1[3] >> 2) & 0x18181818u);
  const unsigned flip = 0x20u << ((code >> 2) * 8);
  const unsigned lo[6] = {d0[0], d0[1], d0[2], d0[3], d1[0], d1[1]};
#pragma unroll
  for (int i = 0; i < 3; ++i) {
    unsigned h = bfi(0x1f1f1f1fu, H[i], H[i] << 1);
    h ^= ((code & 3u) == (unsigned)(i + 1)) ? flip : 0u;
    q[i][0] = __builtin_amdgcn_sbfe((int)perm(h, lo[2 * i], 0x0c040100u), 0, 22);
    q[i][1] = __builtin_amdgcn_sbfe((int)perm(h, lo[2 * i], 0x0c050302u), 0, 22);
    q[i][2] = __builtin_amdgcn_sbfe((int)perm(h, lo[2 * i + 1], 0x0c060100u), 0, 22);
    q[i][3] = __builtin_amdgcn_sbfe((int)perm(h, lo[2 * i + 1], 0x0c070302u), 0, 22);
  }
  return code == 4u;
}

template <int OPS, bool XH = false, typename F>
__device__ __forceinline__ void load_loop(v4u rw, v4u rx, unsigned wv, unsigned xv, int j, unsigned lds0, unsigned k0, int nst, bool pre, F late_hook DBG_PARAM) {
  unsigned so = 0;
  const unsigned wv1 = wv ^ 64;
  const unsigned lb = lds0 + j * 4096, lx = lds0 + 16384 + j * 1024;
  auto issue = [&]() {
    dma16(rw, wv, k0 * NN, lb + so);
    dma16(rw, wv, (k0 + 8) * NN, lb + so + 1024);
    dma16(rw, wv1, (k0 + 16) * NN, lb + so + 2048);
    dma16(rw, wv1, (k0 + 24) * NN, lb + so + 3072);
    const unsigned xk = XH && k0 >= KK / 2 ? k0 + (XR - 1) * (KK / 2) : k0, XS = XH ? KK / 2 : KK;
    dma16_sys(rx, xv, xk, lx + so);
    if (OPS >= 6) dma16_sys(rx, xv, xk + 32 * XS, lx + so + 4096);
    if (OPS >= 7) dma16_sys(rx, xv, xk + 64 * XS, lx + so + 8192);
    k0 = (k0 + 128) & (KK - 1);
    so = so == (LAG + 1) * STAGE ? 0 : so + STAGE;
  };
  if (pre) { k0 = 128; so = STAGE; }
#pragma nounroll
  for (int i = pre; i <= LAG; ++i) issue();
  waitv<LAG * OPS>();
  BARRIER();
#pragma nounroll
  for (int i = 1 + LAG; i < nst; ++i) {
    issue();
    waitv<LAG * OPS>();
    BARRIER();
  }
  late_hook();
  static_assert(LAG == 2, "tail");
  waitv<OPS>(); BARRIER();
  waitv<0>(); BARRIER();
}
struct NoHook { __device__ void operator()() const {} };
template <int MT, bool MID = false, typename F, typename G = NoHook>
__device__ __forceinline__ void gemm_mma(v4i (&acc)[MT][2], int wave, int lane, unsigned lds0, int nst, F hook, G mid = G()) {
  const int g = lane >> 4, r = lane & 15, q = r >> 1;
  unsigned aa[2], ba[2];
#pragma unroll
  for (int kk = 0; kk < 2; ++kk) aa[kk] = lds0 + 16384 + r * 128 + (((4 * kk + g) ^ q) * 16);
#pragma unroll
  for (int n = 0; n < 2; ++n) ba[n] = lds0 + (16 * g + q) * 128 + (((2 * wave + n) ^ ((q >> 1) | ((g & 1) << 2))) * 16) + 8 * (lane & 1);
#pragma unroll
  for (int m = 0; m < MT; ++m) { acc[m][0] = v4i{0, 0, 0, 0}; acc[m][1] = v4i{0, 0, 0, 0}; }
  v4i A[2][2][MT], B[2][2][2];
  unsigned a0, a1, b0, b1;
  auto adr = [&](unsigned so) {
    a0 = aa[0] + so, a1 = aa[1] + so, b0 = ba[0] + so, b1 = ba[1] + so;
    asm volatile("" : "+v"(a0), "+v"(a1), "+v"(b0), "+v"(b1));
  };
  auto rd = [&](int set) {
#pragma unroll
    for (int kk = 0; kk < 2; ++kk) {
#pragma unroll
      for (int n = 0; n < 2; ++n) {
        const v2i lo = __builtin_amdgcn_ds_read_tr8_b64_v2i32((LDS_AS v2i*)(uintptr_t)((n ? b1 : b0) + kk * 8192));
        const v2i hi = __builtin_amdgcn_ds_read_tr8_b64_v2i32((LDS_AS v2i*)(uintptr_t)((n ? b1 : b0) + kk * 8192 + 1024));
        B[set][kk][n] = v4i{lo[0], lo[1], hi[0], hi[1]};
      }
#pragma unroll
      for (int m = 0; m < MT; ++m) A[set][kk][m] = *(LDS_AS v4i*)(uintptr_t)((kk ? a1 : a0) + m * 2048);
    }
  };
  auto mm = [&](int set) {
#pragma unroll
    for (int kk = 0; kk < 2; ++kk)
#pragma unroll
      for (int i = 0; i < 2 * MT; ++i) {
        const int m = i >> 1, n = (i ^ m) & 1;
        acc[m][n] = __builtin_amdgcn_mfma_i32_16x16x64_i8(B[set][kk][n], A[set][kk][m], acc[m][n], 0, 0, 0);
      }
  };
  auto step = [&](int cur, unsigned so) {
    adr(so);
    __builtin_amdgcn_s_waitcnt(0xC07F);
    BARRIER();
    __builtin_amdgcn_sched_barrier(0);
    rd(cur ^ 1);
    mm(cur);
    if (MT == 3) {
      for (int i = 0; i < 7; ++i) {
        __builtin_amdgcn_sched_group_barrier(0x100, 2, 0);
        __builtin_amdgcn_sched_group_barrier(0x008, 1, 0);
      }
      __builtin_amdgcn_sched_group_barrier(0x008, 5, 0);
    } else {
      for (int i = 0; i < 8 + 2 * MT; ++i) {
        __builtin_amdgcn_sched_group_barrier(0x100, 1, 0);
        __builtin_amdgcn_sched_group_barrier(0x008, 1, 0);
      }
      if (MT > 4) __builtin_amdgcn_sched_group_barrier(0x008, 2 * MT - 8, 0);
    }
    __builtin_amdgcn_sched_barrier(0);
  };
  adr(0);
  BARRIER();
  rd(0);
  unsigned so = STAGE;
  auto steps = [&](int n) {
#pragma nounroll
    for (int s = 0; s < n; s += 2) {
      step(0, so);
      so += STAGE;
      so = so == (LAG + 2) * STAGE ? 0 : so;
      step(1, so);
      so += STAGE;
    }
  };
  if (MID) {
    steps(12);
    mid();
    steps(nst - 14);
  } else {
    steps(nst - 2);
  }
  adr(so);
  __builtin_amdgcn_s_waitcnt(0xC07F);
  __builtin_amdgcn_sched_barrier(0);
  mm(0);
  __builtin_amdgcn_sched_barrier(0);
  hook();
  BARRIER();
  rd(1);
  mm(1);
}
template <int MT>
__device__ __forceinline__ void tile_to_lds(const v4i (&acc)[MT][2], int wave, int lane, unsigned lds0) {
  const int g = lane >> 4, r = lane & 15;
#pragma unroll
  for (int m = 0; m < MT; ++m)
#pragma unroll
    for (int n = 0; n < 2; ++n)
      *(LDS_AS v4i*)(uintptr_t)(lds0 + LDS_T + (16 * m + r) * 512 + (((8 * wave + 4 * n + g) ^ (r & 7)) * 16)) = acc[m][n];
  LGKM0;
}
__device__ __forceinline__ v4i tile_quad(int row, int lane, unsigned lds0) {
  return *(LDS_AS v4i*)(uintptr_t)(lds0 + LDS_T + row * 512 + (((lane & 31) ^ (row & 7)) * 16));
}

__device__ __forceinline__ void touch(v4u rsrc, unsigned off) {
  asm volatile("s_mov_b32 m0, %2\n\ts_nop 0\n\tbuffer_load_dword %0, %1, 0 offen lds" :: "v"(off), "s"(rsrc), "n"(LDS_T) : "memory");
}

__global__ void __launch_bounds__(NT, 2) gemm_a2a(Params pa) {
  __shared__ __attribute__((aligned(16))) char lds[LDS_BYTES];
  typedef unsigned v8u __attribute__((ext_vector_type(8)));
  unsigned long long tstart;
  v8u ka, kb, kp; v4u ks, kf;
  int b = blockIdx.x, tid = threadIdx.x;
  {
    unsigned t0, t1;
    asm volatile("s_memrealtime %0\n\ts_load_dwordx8 %1, %8, 0x0\n\ts_and_b32 %5, %7, 7\n\ts_lshl_b32 %5, %5, 5\n\t"
                 "s_load_dwordx8 %3, %8, %5 offset:0x40\n\ts_lshr_b32 %6, %7, 3\n\ts_lshl_b32 %6, %6, 4\n\t"
                 "s_load_dwordx4 %4, %8, %6 offset:0x140\n\ts_load_dwordx8 %2, %8, 0x20"
                 : "=&s"(tstart), "=&s"(ka), "=&s"(kb), "=&s"(kp), "=&s"(ks), "=&s"(t0), "=&s"(t1), "+s"(b)
                 : "s"(__builtin_amdgcn_kernarg_segment_ptr()) : "scc");
    asm volatile("" : "+v"(tid));
    static_assert(offsetof(Params, peer) == 0x40 && offsetof(Params, sched) == 0x140 && offsetof(Params, pf) == 0x340 &&
                  sizeof(Peer) == 32 && sizeof(Sched) == 16, "kernarg layout");
  }
  const int lane = tid & 63;
  const int wave = __builtin_amdgcn_readfirstlane(tid >> 6);
  const int dsel = b & 7, idx = ((b >> 3) - NXW) & 31;
  asm volatile("s_load_dwordx4 %0, %1, %2 offset:0x340" : "=&s"(kf)
               : "s"(__builtin_amdgcn_kernarg_segment_ptr()), "s"(((idx > PFM0 && idx < NWK ? idx - PFM0 : 0) * 8 + wave) * 16));
  const int j = wave & 3;
  unsigned wv = (32 * j + (lane >> 3)) * NN + 16 * ((lane & 7) ^ (lane >> 4));
  unsigned xv = (8 * j + (lane >> 3)) * KK + 16 * ((lane & 7) ^ (lane >> 4) ^ ((j & 1) << 2));
  int role = dsel != 0 ? idx : 32 + idx;
  int early = __builtin_amdgcn_readfirstlane(role < NWK ? (wave >= 4 ? 2 : 0) : role < 32 ? (wave < XPW ? 1 : 0) : role < 46 ? (wave >= 4 ? 3 : 0) : role < 60 ? 4 : 0);
  unsigned xso = ((((idx - NWK) * XPW + wave) >> 1) * KK + (wave & 1) * 1024) + lane * 16;
  asm volatile("" : "+v"(wv), "+v"(xv), "+v"(xso));
  asm volatile("s_waitcnt lgkmcnt(0)" : "+s"(tstart), "+s"(ka), "+s"(kb), "+s"(kp), "+s"(ks), "+s"(kf), "+s"(role), "+s"(early));
  auto ptr = [](unsigned lo, unsigned hi) { return (char*)(((unsigned long long)hi << 32) | lo); };
  struct { const char *x, *w, *sx, *sw; char* myws; char* myout; } p =
      {ptr(ka[0], ka[1]), ptr(ka[2], ka[3]), ptr(ka[4], ka[5]), ptr(ka[6], ka[7]), ptr(kb[0], kb[1]), ptr(kb[2], kb[3])};
  const unsigned lds0 = (unsigned)(uintptr_t)(LDS_AS char*)lds;
  const char* const rws = ptr(kp[0], kp[1]);
  const int rank = kb[4], dst = kp[6];
  const int nsa = kp[7];
  const unsigned t_start = ks[0] & 0xffff, t_raw = ks[0] >> 16, t_2 = ks[1] & 0xffff;
  const char* const cntp = p.myws + WS_CNT + b * 128;
  gu32* const tmo = (gu32*)(p.myws + WS_TMO);
  const v4f nan4 = {__builtin_nanf(""), __builtin_nanf(""), __builtin_nanf(""), __builtin_nanf("")};
  unsigned call = 0;
  float scale = 0.f;
  auto consts_now = [&]() {
    unsigned c;
    float a, d;
    asm volatile("s_load_dword %0, %3, 0x0\n\ts_load_dword %1, %4, 0x0\n\ts_load_dword %2, %5, 0x0\n\ts_waitcnt lgkmcnt(0)"
                 : "=&s"(c), "=&s"(a), "=&s"(d) : "s"(cntp), "s"(p.sx), "s"(p.sw) : "memory");
    call = c + 1u;
    scale = a * d;
  };
  auto consts_lds = [&]() {
    const v4u c = *(volatile LDS_AS v4u*)(uintptr_t)(lds0 + LDS_C);
    call = __builtin_amdgcn_readfirstlane(c[0]) + 1u;
    scale = __uint_as_float(c[1]) * __uint_as_float(c[2]);
    asm volatile("" : "+v"(scale) :: "memory");
  };
  auto wait_until = [&](unsigned ticks) {
    while ((unsigned)__builtin_amdgcn_s_memrealtime() - (unsigned)tstart < ticks) __builtin_amdgcn_s_sleep(1);
  };
  const unsigned row0 = ks[2] & 0xffff, col0 = ks[2] >> 16;
  const char* gw = ptr(kp[4], kp[5]) + col0;
  const char* gx = p.x + row0 * KK;
  int mt = ks[3] & 0xff;
  auto warm_unit = [&](unsigned e) {
    if (e & 0x8000) touch(make_rsrc_s(p.x, ML * KK), (((e >> 3) & 31) * 16 + (lane >> 2)) * KK + (e & 7) * 512 + (lane & 3) * 128);
    else {
      const unsigned row = e & 0x200 ? (e & 31) + (e & 32) * 64 + lane * 32 : (e & 63) * 64 + lane;
      touch(make_rsrc_s(ptr(kp[4], kp[5]), (unsigned)(KK - 1) * NN + NC), row * NN + ((e >> 6) & 3) * 128);
    }
  };
  auto warm_b = [&](int u) {
    const int st = u / 17, k = u - st * 17, k0 = (nsa + st) * 128;
    if (k < 14) touch(make_rsrc_s(p.w, (unsigned)KK * NN), (k0 + (k & 1) * 64 + lane) * NN + (((k >> 1) + 1) ^ rank) * NC);
    else touch(make_rsrc_s(p.x, ML * KK), ((k - 14) * 64 + lane) * KK + k0);
  };
  const char* const bw = p.w + (((idx >> 1) + 1) ^ rank) * NC + (size_t)nsa * 128 * NN;
  const char* const bx = p.x + (idx & 1) * 96 * KK + nsa * 128;
  const bool pre = (early == 2 && t_start == 0) || early == 3;
  if ((unsigned)(role - PFM0) < (unsigned)(NWK - PFM0) && t_start != 0) {
    if ((kf[0] & 0xffff) != 0xffff) warm_unit(kf[0] & 0xffff);
  } else if (early == 4) {
    warm_b(wave * 14 + idx - 14);
  } else if (pre) {
    const int pm = early == 3 ? 6 : mt;
    const v4u rw = make_rsrc_s(early == 3 ? bw : gw, 127u * NN + 128), rx = make_rsrc_s(early == 3 ? bx : gx, (pm * 16 - 1) * KK + 128);
    const unsigned lb = lds0 + j * 4096, lx = lds0 + 16384 + j * 1024;
    dma16(rw, wv, 0, lb);
    dma16(rw, wv, 8 * NN, lb + 1024);
    dma16(rw, wv ^ 64, 16 * NN, lb + 2048);
    dma16(rw, wv ^ 64, 24 * NN, lb + 3072);
    dma16_sys(rx, xv, 0, lx);
    if (j + 4 < 2 * pm) dma16_sys(rx, xv, 32 * KK, lx + 4096);
    if (pm == 6) dma16_sys(rx, xv, 64 * KK, lx + 8192);
  } else if (early == 1) {
    constexpr int NW = XPW * NXW;
    v4u xd[XD];
    const char* const xsrc = p.x + (size_t)NRG * TM * KK;
    auto load = [&](int i) {
      asm volatile("global_load_dwordx4 %0, %1, %2" : "=v"(xd[i % XD])
                   : "v"(xso + (i % (NXP / 2)) * (NW / 2) * KK + (i / (NXP / 2)) * (KK / 2)), "s"(xsrc) : "memory");
    };
#pragma unroll
    for (int i = 0; i < XD; ++i) load(i);
    const int w = (idx - NWK) * XPW + wave;
    const unsigned po = w * 1024 + lane * 16;
    consts_now();
    const char* dstp = rws + SL_XS + (call & 1u) * SL_PAR;
    const char* xflag = rws + SL_XF + w * 128;
#pragma unroll
    for (int i = 0; i < NXP; ++i) {
      constexpr int F = NXP / 2 + XD - 1;
      const int younger = (i < XD ? XD - 1 - i + 2 * i : 2 * (XD - 1)) - (i + XD - 1 >= NXP ? i + XD - NXP : 0) + (i > F && i < F + XD ? 1 : 0);
      asm volatile("s_waitcnt vmcnt(%1)" : "+v"(xd[i % XD]) : "n"(younger) : "memory");
      if (i == 0) STAMPV(1);
      if (i == XD) STAMPV(8);
      if (i == F) STAMPV(9);
      st16(dstp, po + i * NW * 1024, xd[i % XD]);
      if (i == F && lane == 0) st_flag(xflag, 2 * call - 1);
      if (i + XD < NXP) load(i + XD);
    }
    waitv<0>();
    if (lane == 0) st_flag(xflag, 2 * call);
    STAMPV(2);
  }
  const int ownpn = 3 - (dsel >> 1), owncol = rank * NC + ownpn * 128;
  auto warm = [&]() {
    unsigned long long lo = ((unsigned long long)kf[1] << 32) | kf[0], hi = ((unsigned long long)kf[3] << 32) | kf[2];
    for (int r = 1; r < 16; ++r) {
      lo = (lo >> 16) | (hi << 48);
      hi >>= 16;
      if (r == 8) {
        if (idx < PFM1) break;
        const CONST_AS unsigned long long* q = (const CONST_AS unsigned long long*)__builtin_amdgcn_kernarg_segment_ptr() +
                                              (offsetof(Params, pf2) + ((idx - PFM1) * 8 + wave) * 16) / 8;
        lo = q[0]; hi = q[1];
      }
      const unsigned e = lo & 0xffff;
      if (e == 0xffff) break;
      waitv<0>();
      warm_unit(e);
    }
    waitv<0>();
  };

  const char* go = ptr(kp[2], kp[3]) + (row0 * NC + col0) * 4;
  int kind = (ks[3] >> 8) & 0xff;
  const int unit = (ks[3] >> 16) & 0xff;
  const int nst = role < NSPL ? nsa : NST;
  unsigned kfirst = 0;
  if (role >= NWK && role < 32) {
    wait_until(t_start);
  } else if (__builtin_expect(role < NWK, 1)) {
    if (__builtin_expect(t_start > 0, 0)) {
      if (idx >= PFM0) warm();
      STAMPV(7);
      if (mt) wait_until(t_start);
      else {
        consts_now();
        BARRIER();
      }
    }
  } else {
    const int h = idx - 28;
    {
      if (idx >= 14 && h < 0) {
        for (int u = wave * 14 + idx - 14 + 112; u < (NST - nsa) * 17; u += 112) {
          waitv<0>();
          warm_b(u);
        }
        waitv<0>();
        STAMPV(1);
      } else if (h < 0) {
        const int d = (idx >> 1) + 1, t = idx & 1;
        if (wave >= 4) {
          const v4u rw = make_rsrc_s(bw, (unsigned)(KK - nsa * 128 - 1) * NN + 128), rx = make_rsrc_s(bx, 96 * KK - nsa * 128);
          load_loop<7>(rw, rx, wv, xv, j, lds0, 0u, NST - nsa, true, []() {} DBG_ARG);
        } else {
          v4i acc[6][2];
          consts_now();
          gemm_mma<6>(acc, wave, lane, lds0, NST - nsa, []() {});
          asm volatile("s_nop 9" : "+v"(acc[0][0]), "+v"(acc[0][1]), "+v"(acc[1][0]), "+v"(acc[1][1]), "+v"(acc[2][0]), "+v"(acc[2][1]),
                       "+v"(acc[3][0]), "+v"(acc[3][1]), "+v"(acc[4][0]), "+v"(acc[4][1]), "+v"(acc[5][0]), "+v"(acc[5][1]));
          STAMPV(3);
          const char* pb = p.myws + WS_PB + (size_t)((d * NSPL + 2 * t) * 4 + wave) * 6144 + lane * 16;
          const char* bf = p.myws + WS_BF + ((d * NSPL + 2 * t) * 4 + wave) * 128;
#pragma unroll
          for (int m = 0; m < 6; ++m) {
            st16(pb, (m / 3) * 4 * 6144 + (m % 3) * 2048, acc[m][0]);
            st16(pb, (m / 3) * 4 * 6144 + (m % 3) * 2048 + 1024, acc[m][1]);
          }
          waitv<6>();
          if (lane == 0) st_flag(bf, call);
          STAMPV(8);
          waitv<1>();
          if (lane == 0) st_flag(bf + 4 * 128, call);
          STAMPV(5);
        }
      }
    }
    if (h >= 0) {
      wait_until(t_2);
      kind = 2;
    } else {
      consts_now();
      const v4u rw = make_rsrc(p.w, (unsigned)KK * NN);
      if (idx >= 14) {
        wait_until(t_2);
        for (int u = wave * 14 + idx - 14; u < 256; u += 112) {
          touch(rw, ((u >> 2) + lane * 64) * NN + rank * NC + (u & 3) * 128);
          waitv<0>();
        }
      }
      STAMPV(2);
      const int js = idx >> 2;
      if (wave == 4 && lane < XPW * NXW) poll_ge(p.myws + (js + 1) * SLAB + SL_XF + lane * 128, 2 * call - 1, tmo, call);
      wait_until(kb[6]);
      BARRIER();
      STAMPV(10);
      gw = p.w + rank * NC + (idx & 3) * 128;
      gx = p.myws + (js + 1) * SLAB + SL_XS + (call & 1) * SL_PAR;
      go = p.myout + ((size_t)((rank ^ (js + 1)) * ML + NRG * TM) * NC + (idx & 3) * 128) * 4;
      mt = 5;
      kind = 3;
    }
  }
  if (__builtin_expect(kind == 2, 0)) {
    const int h = idx - 28, r0 = (~dsel & 1) * 256 + h * 64;
    gw = p.w + owncol;
    gx = p.x + r0 * KK;
    go = p.myout + ((size_t)(rank * ML + r0) * NC + ownpn * 128) * 4;
    mt = 4;
    kfirst = h * 1024;
  }

  if (__builtin_expect(mt != 0, 1)) {
    if (__builtin_expect(wave >= 4, 1)) {
      const v4u rw = make_rsrc(gw, (unsigned)(KK - 1) * NN + 128), rx = make_rsrc(gx, mt * 16 * KK);
      const bool go_on = pre && role < NWK;
      if (kind == 3) {
        const unsigned xh = (xv & 127) | ((xv >> 12) << 11);
        if (j < 2) load_loop<7, true>(rw, rx, wv, xh, j, lds0, 0u, NST, false, consts_lds DBG_ARG);
        else load_loop<6, true>(rw, rx, wv, xh, j, lds0, 0u, NST, false, consts_lds DBG_ARG);
      } else if (mt == 6) load_loop<7>(rw, rx, wv, xv, j, lds0, 0u, NST, go_on, consts_lds DBG_ARG);
      else if (j + 4 < 2 * mt) load_loop<6>(rw, rx, wv, xv, j, lds0, kfirst, nst, go_on, consts_lds DBG_ARG);
      else load_loop<5>(rw, rx, wv, xv, j, lds0, 0u, nst, go_on, consts_lds DBG_ARG);
    } else {
      if (wave == 0) {
        if (lane < 3) {
          const void* src = lane == 0 ? cntp : lane == 1 ? p.sx : p.sw;
          asm volatile("s_mov_b32 m0, %1\n\ts_nop 0\n\tglobal_load_lds_dword %0, off sc1" :: "v"(src), "s"(lds0 + LDS_C) : "memory");
        }
        waitv<0>();
      }
      auto send = [&](const v4i& q0, const v4i& q1, const v4i& q2, int half, int n) {
        const char* const rpk = rws + SL_PK + (call & 1) * SL_PAR + (size_t)(unit + half) * PK_TILE + wave * 4096 + lane * 16;
        const v4i q[3] = {q0, q1, q2};
        st16(rpk, n * 2048, pack12_lo(q));
        v4u d1;
        const bool esc = pack12_hi(q, d1);
        st16(rpk, n * 2048 + 1024, d1);
        if (esc) {
          const unsigned eo = ((half * TM + (lane & 15)) * NC + 32 * wave + 16 * n + 4 * (lane >> 4)) * 4;
#pragma unroll
          for (int m = 0; m < 3; ++m) st16(go, eo + 16 * m * NC * 4, dequant(q[m], scale));
        }
      };
      if (__builtin_expect(mt == 3, 1)) {
        v4i acc[3][2];
        bool lost = false;
        const unsigned lp = lds0 + LDS_T + 1024 + wave * 6144;
        gemm_mma<3>(acc, wave, lane, lds0, nst, [&]() {
          consts_lds();
          if (nst == NST) return;
          STAMPV(9);
          int ok = 1;
          if (lane == 0) ok = poll_ge(p.myws + WS_BF + ((dsel * NSPL + unit) * 4 + wave) * 128, call, tmo, call);
          STAMPV(10);
          lost = !__builtin_amdgcn_readfirstlane(ok);
          if (lost) return;
          const v4u rb = make_rsrc(p.myws + WS_PB + (size_t)((dsel * NSPL + unit) * 4 + wave) * 6144, 6144);
#pragma unroll
          for (int i = 0; i < 6; ++i) dma16_sys(rb, lane * 16, i * 1024, lp + i * 1024);
        });
        if (nst != NST) {
          waitv<0>();
#pragma unroll
          for (int i = 0; i < 6; ++i) acc[i >> 1][i & 1] += *(LDS_AS v4i*)(uintptr_t)(lp + i * 1024 + lane * 16);
          if (lost) {
            scale = __builtin_nanf("");
            acc[0][0][0] = acc[0][1][0] = 0x7fffffff;
          }
        }
        STAMPV(3);
        if (__builtin_expect(kind, 0)) tile_to_lds<3>(acc, wave, lane, lds0);
        else {
          send(acc[0][0], acc[1][0], acc[2][0], 0, 0);
          send(acc[0][1], acc[1][1], acc[2][1], 0, 1);
        }
      } else if (mt == 6) {
        v4i acc[6][2];
        gemm_mma<6>(acc, wave, lane, lds0, NST, consts_lds);
        STAMPV(3);
        if (kind != 1) {
          send(acc[0][0], acc[1][0], acc[2][0], 0, 0);
          send(acc[0][1], acc[1][1], acc[2][1], 0, 1);
        }
        if (kind) tile_to_lds<6>(acc, wave, lane, lds0);
        else {
          send(acc[3][0], acc[4][0], acc[5][0], 1, 0);
          send(acc[3][1], acc[4][1], acc[5][1], 1, 1);
        }
      } else if (mt == 5) {
        v4i acc[5][2];
        gemm_mma<5, true>(acc, wave, lane, lds0, NST, consts_lds, [&]() {
          if (wave == 0 && lane < XPW * NXW) poll_ge(p.myws + ((idx >> 2) + 1) * SLAB + SL_XF + lane * 128, 2 * call, tmo, call);
        });
        tile_to_lds<5>(acc, wave, lane, lds0);
      } else {
        v4i acc[4][2];
        gemm_mma<4>(acc, wave, lane, lds0, NST, consts_lds);
        tile_to_lds<4>(acc, wave, lane, lds0);
      }
      if (kind == 0) {
        STAMPV(4);
        waitv<0>();
        if (lane == 0) st_flag(rws + SL_TF + (unit * 4 + wave) * 128, call);
        STAMPV(5);
      }
    }
    if (kind) {
      BARRIER();
      if (kind == 2 || kind == 3) STAMPV(11);
      const bool bad = kind == 3 && __hip_atomic_load(tmo, __ATOMIC_RELAXED, __HIP_MEMORY_SCOPE_AGENT) == call;
      if (kind == 5 && wave < 4) {
        waitv<0>();
        if (lane == 0) st_flag(rws + SL_TF + (unit * 4 + wave) * 128, call);
        STAMPV(4);
      } else {
        const int r0 = (kind == 5 ? TM - 8 : 0) + 2 * wave + (lane >> 5), rs = kind == 5 ? 8 : 16;
        v4f y[6];
#pragma unroll
        for (int i = 0; i < 6; ++i) {
          y[i] = dequant(tile_quad(r0 + rs * i, lane, lds0), scale);
          if (bad) y[i] = nan4;
        }
        const unsigned o = (r0 * NC + 4 * (lane & 31)) * 4;
        if ((kind & 3) == 1) { wait_until(t_raw); STAMPV(4); }
#pragma unroll
        for (int i = 0; i < 6; ++i)
          if (i < mt) st16(go, o + rs * i * NC * 4, y[i]);
      }
      if ((kind & 3) == 1) {
        waitv<0>();
        BARRIER();
        if (tid == 0) st_flag(rws + SL_TF + (unit + (kind >> 2)) * 4 * 128, call);
        STAMPV(5);
      }
    }
  }
  const int umem = ks[3] >> 24;
  if (role < NWK && umem != 0xff) {
    call = __builtin_amdgcn_readfirstlane(call);
    const int nmem = kb[5];
    const v4u rp = make_rsrc(p.myws + dsel * SLAB + SL_PK + (call & 1) * SL_PAR, NTILE * PK_TILE);
    const unsigned lw = lds0 + LDS_T + 1024 + (wave < 4 ? wave * 6144 : 6144 + wave * 4608);
    const unsigned fo = (lane >> 5) * 2048 + ((lane >> 3) & 3) * 256 + (lane & 7) * 16;
    const unsigned la = lw + (lane & 7) * 128 + (lane >> 3) * 16;
    const unsigned lo = ((lane >> 3) * NC + 4 * (lane & 7)) * 4;
#pragma nounroll
    for (int t = wave * nmem + umem; t < 8 * NTILE; t += 8 * nmem) {
      const unsigned ui = pa.uinfo[t >> 3];
      const bool raw = ui & 128;
      if (raw && ((ui & 64) || (t & 7))) continue;
      const int fi = (t >> 1) - (ui & 64 ? 4 : 0);
      const char* o = p.myout + (size_t)((dst * ML + (ui & 15) * TM + 8 * (t & 1)) * NC + ((ui >> 4) & 3) * 128 + 16 * (t & 6)) * 4 + lo;
      const char *o1 = o + 16 * NC * 4, *o2 = o + 32 * NC * 4;
      asm volatile("" : "+v"(o), "+v"(o1), "+v"(o2));
      int ok = 1;
      if (lane == 0) ok = poll_ge(p.myws + dsel * SLAB + SL_TF + fi * 128, call, tmo, call);
      ok = __builtin_amdgcn_readfirstlane(ok);
      if (raw) {
        if (!ok && lane == 0) st_flag(p.myout + (size_t)rank * ML * NC * 4, 0x7fc00000u);
      } else if (ok) {
        dma16_sys(rp, fo, (t >> 1) * 4096 + (t & 1) * 128, lw);
        dma16_sys(rp, fo, (t >> 1) * 4096 + (t & 1) * 128 + 1024, lw + 1024);
        waitv<0>();
        const v4u d0 = *(LDS_AS v4u*)(uintptr_t)la, d1 = *(LDS_AS v4u*)(uintptr_t)(la + 1024);
        v4i q[3];
        if (!unpack12(d0, d1, q)) {
          st16(o, 0, dequant(q[0], scale));
          st16(o1, 0, dequant(q[1], scale));
          st16(o2, 0, dequant(q[2], scale));
        }
      } else {
        st16(o, 0, nan4);
        st16(o1, 0, nan4);
        st16(o2, 0, nan4);
      }
    }
  }
  if (tid == 0) __hip_atomic_store((gu32*)cntp, call, __ATOMIC_RELAXED, __HIP_MEMORY_SCOPE_AGENT);
}

struct Tune { int c0, slope, tallx, traw; unsigned pfcnt; int pfskip, pfdelay, town, rho, lpf, lown, flags, taus, taut, nsa, lay, nuh, lgo, fast, mixd, adj[28]; };
static const Tune g_tune =
    {3, 29, 240, 1600, 0xcccccccc, 0, 50, 900, 120, 300, 50, 80, 185, 250, 22, 4, 9, 720, 60, 0, {}};
static const char* const g_layout[] = {
    "00s01s02s03s04s05s06s07s08s10s11t13t15t17t20s21t23t25t27t31t33t30s35T37T",
    "00s01s02s03s04s05s06s07s08s10s11t13t15t17t20s21t23t25t27t31t33t30s35m37m",
    "00s01s02s03s04s05s06s07s08s10s11t13t15t17t20s21t23t25t27t31t33t35m37m30s",
    "00s01s02s03s04s05s06s07s08s10s11t13t15t17t20s21t23t25t27t31t33t35m37m30S",
    "00s01s02s03s04s05s06s07s08s10s11t13t15t17t20s21t23t25t27t31t33t30s35m37T",
    "00s01s02s03s04s05s06s07s08s10s11t13t15t17t20s21t23t25t27t31t33m35m37m30s",
    "00s01s02s03s04s05s06s07s08s10s20s11t13t15t17t21t23t25t27t31t33t30s35m37T",
    "00s01s02s03s04s05s06s07s08s10s11s12s13t15t17t20s21t23t25t27t31t33t30s35m37T",
    "00s01s02s03s04s05s06s07s08s10s11t13t15t17t21t23t20s25t27t31t33t30s35m37T",
};

static void make_schedule(Params& p) {
  const Tune& t = g_tune;
  const char* lay = g_layout[t.lay];
  int start[NWK], pnl[NWK], rgs[NWK], tall[NWK], own[NWK], ncmp = 0, unit = 0, nmix = 0;
  for (int i = 0; i < 32; ++i) {
    Sched& s = p.sched[(i + NXW) & 31];
    s.start = t.town; s.traw = t.traw; s.t2 = i < 28 ? t.lpf : t.lown; s.pad = 0;
    s.row0 = s.col0 = 0; s.mt = s.kind = s.unit = 0; s.umem = 0xff;
    if (i >= 28) { s.mt = 4; s.kind = 2; if (i < NWK) start[i] = t.town; continue; }
    start[i] = 30000;
    if (!*lay) continue;
    int pn = lay[0] - '0';
    if ((t.flags & 64) && pn >= 2) pn = 5 - pn;
    const int rg = lay[1] - '0', mix = lay[2] == 'm', tl = (lay[2] | 32) == 't' || mix, raw = !(lay[2] & 32);
    lay += 3;
    own[i] = i < NSPL ? t.nsa : NST;
    const int st = i < NSPL ? 0 : (unit - nmix - t.c0 + 1) * t.slope - (tl ? t.tallx : 0) + (mix ? t.mixd : 0) + t.adj[i];
    nmix += mix;
    start[i] = s.start = st > t.fast ? st : 0; pnl[i] = pn; rgs[i] = rg; tall[i] = tl;
    s.row0 = rg * TM; s.col0 = pn * 128; s.mt = tl ? 6 : 3; s.unit = unit; s.kind = mix ? 5 : raw;
    p.uinfo[unit++] = rg | pn << 4 | raw << 7;
    if (tl) p.uinfo[unit++] = (rg + 1) | pn << 4 | (mix ? 128 : 64 | raw << 7);
    ncmp = i + 1;
  }
  for (int i = 0; i < NR; ++i) p.peer[i].nsa = t.nsa;
  int nmem = 0;
  for (int i = ncmp; i < 28; ++i) p.sched[i + NXW].umem = nmem++;
  for (int i = 0; i < t.nuh; ++i) p.sched[i + NXW].umem = nmem++;
  p.nmem = nmem; p.pad = 0; p.lgo = t.lgo;
  struct U { int need; unsigned short e; } u[4 * 64 + 27 * 8];
  int n = 0;
  auto need = [&](int i, int st10) {
    if (st10 >= own[i] * 10) return 1 << 30;
    return (start[i] > 50 ? start[i] : 50) + 20 + (st10 - 20) * (tall[i] ? t.taut : t.taus) / 100;
  };
  for (int pn = 0; pn < 4; ++pn)
    for (int h = 0; h < 64; ++h) {
      int m = 1 << 30;
      const bool far = pn == 3 && (t.flags & 16);
      const int st10 = far ? (h & 32) * 5 : h * 5;
      for (int i = 0; i < ncmp; ++i) if (pnl[i] == pn && need(i, st10) < m) m = need(i, st10);
      u[n++] = {m, (unsigned short)(pn << 6 | h | (far ? 0x200 : 0))};
    }
  for (int rb = 0; rb < 27; ++rb)
    for (int q = 0; q < 8; ++q) {
      int m = 1 << 30;
      for (int i = 0; i < ncmp; ++i) if ((rb / 3 == rgs[i] || (tall[i] && rb / 3 == rgs[i] + 1)) && need(i, q * 40 + 10) < m) m = need(i, q * 40 + 10);
      u[n++] = {m, (unsigned short)(0x8000 | rb << 3 | q)};
    }
  for (int i = 1; i < n; ++i) { const U v = u[i]; int k = i; while (k > 0 && u[k - 1].need > v.need) { u[k] = u[k - 1]; --k; } u[k] = v; }
  int next = 0, slack = 1 << 30;
  while (next < n && u[next].need < t.pfskip) ++next;
  int used[32] = {}, order[NWK], no = 0;
  for (int m = PFM0; m < NWK; ++m)
    for (int w = 0; w < 8; ++w)
      for (int r = 0; r < 8; ++r) { p.pf[m - PFM0][w][r] = 0xffff; if (m >= PFM1 && m < NWK) p.pf2[m - PFM1][w][r] = 0xffff; }
  for (int i = PFM0; i < NWK; ++i) { int k = no++; while (k > 0 && start[order[k - 1]] < start[i]) { order[k] = order[k - 1]; --k; } order[k] = i; }
  for (int r = 0; r < 16; ++r) {
    const int cnt = (t.pfcnt >> (4 * (r < 7 ? r : 7))) & 15;
    int mem[32], nm = 0;
    for (int k = 0; k < no && k < cnt; ++k) {
      const int i = order[k];
      if (used[i] < (i >= PFM1 ? 16 : 8) && t.pfdelay + (used[i] + 1) * t.rho <= start[i] + 20) mem[nm++] = i;
    }
    for (int w = 0; w < 8; ++w)
      for (int k = 0; k < nm; ++k)
        if (next < n) {
          const int i = mem[k];
          if (r > 0 && u[next].need - (60 + (r + 1) * t.rho) < slack) slack = u[next].need - (60 + (r + 1) * t.rho);
          (used[i] < 8 ? p.pf[i - PFM0][w][used[i]] : p.pf2[i - PFM1][w][used[i] - 8]) = u[next++].e;
        }
    for (int k = 0; k < nm; ++k) ++used[mem[k]];
  }
}

extern "C" void kernel_launch(int rank, int nranks, ncclComm_t comm, hipStream_t stream,
                              void* const* d_in, const int* in_sizes, int n_in, void* d_out, int out_size,
                              const int* in_dtypes, int out_dtype, void* const* d_out_all,
                              void* const* d_ws_all, size_t ws_bytes) {
  if (nranks != NR || n_in != 4 || (unsigned)rank >= (unsigned)NR || in_sizes[0] != ML * KK || in_sizes[1] != KK * NN ||
      out_size != NR * ML * NC || ws_bytes < WS_END) {
    fprintf(stderr, "kernel_launch: unexpected problem shape\n");
    return;
  }
  Params p = {};
  p.x = (const int8_t*)d_in[0];
  p.w = (const int8_t*)d_in[1];
  p.sx = (const float*)d_in[2];
  p.sw = (const float*)d_in[3];
  p.myws = (char*)d_ws_all[rank];
  p.myout = (float*)d_out;
  for (int i = 0; i < NR; ++i) {
    Peer& q = p.peer[i];
    q.dst = i ^ rank;
    q.ws = (char*)d_ws_all[q.dst] + i * SLAB;
    q.out = (char*)d_out_all[q.dst] + (size_t)rank * ML * NC * 4;
    q.w = (const char*)d_in[1] + q.dst * NC;
  }
  p.rank = rank;
  make_schedule(p);
  gemm_a2a<<<NWG, NT, 0, stream>>>(p);
}
```
